```python
import numpy as np

kernel_name = "gpu_gemm_ar_m4096_k4096_n32768_f32_silu"


ABSMAX_RTOL = 0.001

COMPUTE_DTYPE = "float32"
EPILOGUE = "silu"
IN_QUANT = None
OUT_QUANT = None
IN_DTYPES = {"x": "float32", "w_mat": "float32"}
WORLD_SIZE = 8

def setup_inputs():

    x = np.random.randn(4096, 4096).astype(np.float32)
    w_mat = (np.random.randn(4096, 32768) / np.sqrt(4096)).astype(np.float32)
    return {"x": x, "w_mat": w_mat}

def _epilogue(y):
    return (y / (1.0 + np.exp(-np.clip(y, -60.0, 60.0)))).astype(np.float32)

def reference(x, w_mat):
    y = (x @ w_mat).astype(np.float32)
    return _epilogue(y)
```

```cpp
#include <hip/hip_runtime.h>
#include <rccl/rccl.h>
#include <cstdio>
#include <utility>

#define HIP_CHECK(expr) do { hipError_t _e = (expr); if (_e != hipSuccess) { \
    fprintf(stderr, "HIP error %s at %s:%d\n", hipGetErrorString(_e), __FILE__, __LINE__); \
    return; } } while (0)

namespace gk {
typedef _Float16 f16x8 __attribute__((ext_vector_type(8)));
typedef _Float16 f16x2 __attribute__((ext_vector_type(2)));
typedef float f32x4 __attribute__((ext_vector_type(4)));
typedef float f32x2 __attribute__((ext_vector_type(2)));
typedef unsigned u32x4 __attribute__((ext_vector_type(4)));
#define LAS __attribute__((address_space(3)))
#define GAS __attribute__((address_space(1)))
#define DI __device__ __forceinline__

constexpr int M = 4096, N = 32768, NR = 8, KS = 512;
constexpr int NCOMM = 24;
constexpr int OWNS = 16, OWNF = 10, NOPT = OWNS - OWNF;
constexpr int GRID = 256, NG = GRID - NCOMM, HB = NG / 2;
constexpr int NREP = 512 - NR * OWNS;
constexpr int VO = OWNS + NREP;
constexpr int nvs(int l) { return VO + 7 * (NOPT - l); }
constexpr int nv(int l) { return (nvs(l) + 1) / 2; }
constexpr int NST = 4, B0 = HB / 8, BYS = NST * HB / 8;
constexpr int rx(int l, int b) { return (nv(l) - 3 * b) / 2 * 8; }
constexpr int by(int l) {
  int b = BYS + ((nv(l) - 3 * BYS) & 1);
  while ((b - BYS) * 8 + rx(l, b) > HB && 3 * (b + 2) <= nv(l) && (b + 2 - BYS) * 8 + rx(l, b + 2) / 2 <= HB) b += 2;
  return b;
}
static_assert(NCOMM % 8 == 0 && NCOMM <= 32 && 2 * HB % 8 == 0 && B0 == 14 && VO % 2 == 0 && NOPT <= 7 && OWNS <= 18 && 3 * by(NOPT) <= nv(NOPT), "layout");
constexpr int OWNB = (OWNS + 5) / 6;
constexpr int sb_end(int i) { return i == 5 ? VO + 7 * NOPT : i == 4 ? VO : 6 * (((i + 1) * HB + 7) / 8); }
constexpr int sb_beg(int i) { return i == 0 ? 0 : sb_end(i - 1); }
constexpr int sb_units(int i) { return sb_end(i) - sb_beg(i) + (i == 0 ? 64 : 0); }
constexpr int sb_base(int i) { return i == 0 ? 0 : sb_base(i - 1) + 16 * sb_units(i - 1); }
constexpr int NUNITS = sb_base(6), NB = NUNITS / 4, NBO = sb_base(5) / 4;
static_assert(NST == 4 && sb_end(3) < VO, "superblocks");
template <int V> constexpr int C_ = V;
constexpr int S0 = sb_end(0) - OWNS, CV1 = 64 + NR * OWNS + S0, CV2 = NREP - S0, NCV = 16 * (CV1 + CV2), PER = NG / 8, J1 = 4 * NG / CV1;
constexpr unsigned EXO = 16 * 576, EX_FLAGS = (EXO + 256 * OWNS) * 4096, EX_BYTES = EX_FLAGS + (1u << 20);

constexpr unsigned HTB = 16384, TILEB = 32768;
constexpr size_t APANEL = 4 * TILEB, BPANEL = 4 * HTB, ACH = 16 * APANEL, BCH = 256 * BPANEL;
constexpr size_t CTRL = 1 << 20, XA_BYTES = 16 * ACH, WB_OFF = XA_BYTES, EX_OFF = WB_OFF + 16 * BCH, BUF_BYTES = EX_OFF + EX_BYTES;
constexpr unsigned P1 = sb_end(0) / 2, P2 = sb_end(1) / 2, P3 = sb_end(2) / 2, P4 = sb_end(3) / 2, P5 = VO / 2, Q1 = P1 + (P3 - P2), Q5 = Q1 + (P2 - P1);
static_assert(P4 - P3 <= 3 * B0 && P5 - P4 <= P2 - P1 && Q5 + nv(0) - P5 <= 256, "places");
constexpr unsigned OFF_TICKET = 2048, OFF_TMO = 256, OFF_TILEQ = 768, OFF_BATQ = 1280, OFF_PACE = 1536, OFF_DONE = 1792, OFF_PROG = 4096, OFF_RDY = 8192;
constexpr unsigned LDS_BYTES = 163840, LDS_WORD = 65536 + 5 * HTB;
constexpr unsigned STAGGER = 80, TSTEP = 178;
constexpr int SKIP = 4;
constexpr unsigned TH0 = 1600, TH2 = 1666, THD = 39;
constexpr unsigned long long DEADLINE = 100000000ull;

DI unsigned a_off(unsigned pm) { return pm * (unsigned)APANEL; }
DI unsigned b_off(unsigned vp) { return (unsigned)WB_OFF + (vp < P1 ? vp : vp < P2 ? vp - P1 + Q1 : vp < P3 ? vp - P2 + P1 : vp < P4 ? vp - P3 : vp - P5 + Q5) * (unsigned)BPANEL; }
DI unsigned real_strip(unsigned vs, unsigned rank) { const unsigned i = vs - VO, g = i / 7;
  return vs < (unsigned)OWNS ? NREP + OWNS * rank + vs : vs < (unsigned)VO ? vs - OWNS : NREP + OWNS * ((rank + 1 + i - g * 7) & 7u) + OWNS - 1 - g; }
DI unsigned koff_a(unsigned k) { return (k >> 2) * (unsigned)ACH + (k & 3u) * TILEB; }
DI unsigned koff_b(unsigned k) { return (k >> 2) * (unsigned)BCH + (k & 3u) * HTB; }
struct Params { const float* x; const float* w; char* out[NR]; char* ws[NR]; int rank; int pad; };

DI unsigned pk16(float a, float b) { f32x2 v = {a, b}; return __builtin_bit_cast(unsigned, __builtin_convertvector(v, f16x2)); }
DI unsigned rfl(unsigned v) { return __builtin_amdgcn_readfirstlane(v); }
DI int lane_id() { int l; asm volatile("v_mbcnt_lo_u32_b32 %0, -1, 0\n\tv_mbcnt_hi_u32_b32 %0, -1, %0" : "=v"(l)); return l; }
DI unsigned long long rtime() { unsigned long long t; asm volatile("s_memrealtime %0\n\ts_waitcnt lgkmcnt(0)" : "=s"(t) :: "memory"); return t; }
DI char* uptr(const char* q) {
  const unsigned long long b = (unsigned long long)q;
  char* r = (char*)(((unsigned long long)rfl((unsigned)(b >> 32)) << 32) | rfl((unsigned)b));
  asm volatile("s_nop 4" : "+s"(r));
  return r;
}
DI char* upick(char* const (&t)[NR], unsigned q) { return uptr(t[rfl(q) & 7u]); }
DI char* pick(char* const (&t)[NR], unsigned q) { char* b = t[0];
#pragma unroll
  for (int e = 1; e < NR; ++e) b = (q == (unsigned)e) ? t[e] : b;
  return b; }
#define WG_BAR() do { asm volatile("" ::: "memory"); __builtin_amdgcn_s_barrier(); asm volatile("" ::: "memory"); } while (0)

DI void code_prefetch(unsigned& junk, unsigned voff) {
  unsigned long long pc; int d; unsigned n;
  asm volatile("s_getpc_b64 %0\n2:\n\ts_mov_b32 %1, 2b-.Lgk_code_begin\n\ts_mov_b32 %2, .Lgk_code_end-.Lgk_code_begin" : "=s"(pc), "=s"(d), "=s"(n));
  const unsigned long long b = pc - d;
  asm volatile("" : "+v"(voff));
  const unsigned o = (voff >> 12) * 8192 + ((voff >> 4) & 63u) * 128;
#pragma unroll
  for (int i = 0; i < 4; ++i) asm volatile("global_load_dword %0, %1, %2" : "+v"(junk) : "v"(min(o + i * 32768, n)), "s"(b) : "memory");
}
constexpr unsigned QSH = 12;
static_assert(NB + 64 * OWNS + 3 * NCOMM < 4096, "tickets per call");
DI void queue_open(const char* ws, unsigned off, unsigned call) {
  asm volatile("s_mov_b64 exec, 1\n\tglobal_atomic_umax %0, %1, %2\n\ts_mov_b64 exec, -1" :: "v"(off), "v"(call << QSH), "s"(ws) : "memory");
}

constexpr unsigned long long L_PROG = ~(~0ull << NCOMM), L_LEVEL = 1ull << 32, L_FWD = 0xffull << 40, L_RAW = 0xffull << 48;
DI bool all_ge(unsigned w, unsigned long long lanes, unsigned target) { return (__builtin_amdgcn_ballot_w64(w >= target) & lanes) == lanes; }
DI bool prog_ge(unsigned pw, unsigned call, unsigned need) { return all_ge(pw, L_PROG, (call - 1) * NB + need); }
DI unsigned level_of(unsigned w) { return min(__builtin_amdgcn_readlane(w, 32) & 7u, (unsigned)NOPT); }
DI bool wait_words(const char* ws, unsigned call, unsigned off, unsigned stride, unsigned long long lanes, unsigned target, unsigned* seen = nullptr) {
  const unsigned po = off + lane_id() * stride;
  GAS unsigned* tmo = (GAS unsigned*)(ws + OFF_TMO);
  const unsigned long long t0 = rtime();
  for (;;) {
    unsigned w; asm volatile("global_load_dword %0, %1, %2 sc0 sc1\n\ts_waitcnt vmcnt(0)" : "=&v"(w) : "v"(po), "s"(ws) : "memory");
    if (seen) *seen = w;
    if (all_ge(w, lanes, target)) return true;
    if (rfl(__hip_atomic_load(tmo, __ATOMIC_RELAXED, __HIP_MEMORY_SCOPE_AGENT)) == call || rtime() - t0 > DEADLINE) {
      __hip_atomic_store(tmo, call, __ATOMIC_RELAXED, __HIP_MEMORY_SCOPE_AGENT);
      return false; }
    __builtin_amdgcn_s_sleep(8);
  }
}
DI bool wait_progress(const char* ws, unsigned call, unsigned need) { return wait_words(ws, call, OFF_PROG, 16, L_PROG, (call - 1) * NB + need); }
DI bool wait_level(const char* ws, unsigned call, unsigned& level) { unsigned w; const bool ok = wait_words(ws, call, OFF_PROG, 16, L_LEVEL, call * 8, &w); level = level_of(w); return ok; }
DI bool wait_ready(const char* ws, unsigned call, unsigned stage, bool& raw) {
  const bool ok = wait_words(ws, call, OFF_RDY + stage * 8192, 128, ~0ull, call * (stage < 2 ? PER : 8 * OWNB));
  unsigned w; wait_words(ws, call, OFF_PROG, 16, 0ull, 0u, &w);
  raw = (__builtin_amdgcn_ballot_w64(w >= call) & L_RAW) != 0;
  return ok;
}
DI unsigned get_call(const char* ws, LAS unsigned char* lds, unsigned bid, int wid, bool gate) {
  if (wid == 0) {
    unsigned tk = 1u;
    asm volatile("s_mov_b64 exec, 1\n\tglobal_atomic_add %0, %1, %0, %2 sc0\n\ts_mov_b64 exec, -1\n\ts_waitcnt vmcnt(0)"
                 : "+v"(tk) : "v"(OFF_TICKET + (bid & 7u) * 128u), "s"(ws) : "memory");
    const unsigned call = rfl(tk) / (GRID / 8) + 1u;
    if (gate) wait_words(ws, call, OFF_PROG, 16, L_FWD, (call - 1) * NCOMM);
    *(volatile LAS unsigned*)(lds + LDS_WORD) = call;
    asm volatile("s_waitcnt lgkmcnt(0)" ::: "memory");
  }
  WG_BAR();
  return rfl(*(volatile LAS unsigned*)(lds + LDS_WORD));
}
#define TELL(OP, addr, val) asm volatile("s_mov_b64 exec, 0xff\n\t" OP " %0, %1, off sc1\n\ts_mov_b64 exec, -1" :: "v"(addr), "v"(val) : "memory")

struct UnitAddr { const char* src; unsigned slot, own; bool isx; };
DI UnitAddr cv_unit(const Params& p, unsigned u) {
  UnitAddr a; const unsigned r = (unsigned)p.rank; unsigned ch, e, ns, vs = ~0u;
  if (u < 16u * CV1) {
    ch = u / CV1; e = u - ch * CV1;
    if (e < 64u) {
      a.isx = true; a.src = (const char*)p.x + ((size_t)(e * 64u) * KS + ch * 32u) * 4u; a.slot = ch * 576u + e;
      a.own = (unsigned)CTRL + a_off(e >> 2) + ch * (unsigned)ACH + (r >> 1) * TILEB + (e & 3u) * 8192u + (r & 1u) * 1024u;
      return a;
    }
    e -= 64u;
    if (e < NR * OWNS) { ns = NREP + e; const unsigned o = e / OWNS, i = e - o * OWNS;
      if (o == r) vs = i; else if (i >= (unsigned)OWNF) vs = VO + (OWNS - 1 - i) * 7 + ((o - r - 1) & 7u); }
    else { ns = e - NR * OWNS; vs = ns + OWNS; }
  } else { u -= 16u * CV1; ch = u / CV2; ns = S0 + u - ch * CV2; if (ns + OWNS < (unsigned)C_<sb_beg(3)>) vs = ns + OWNS; }
  a.isx = false; a.src = (const char*)p.w + ((size_t)(ch * 32u) * N + ns * 64u) * 4u; a.slot = ch * 576u + 64u + ns;
  a.own = vs == ~0u ? 0u : (unsigned)CTRL + b_off(vs >> 1) + ch * (unsigned)BCH + (r >> 1) * HTB + (vs & 1u) * 8192u + (r & 1u) * 1024u;
  return a;
}
typedef unsigned u32x3 __attribute__((ext_vector_type(3)));
#define LD16(dst, voff, sbase) asm volatile("global_load_dwordx4 %0, %1, %2 nt" : "=v"(dst) : "v"(voff), "s"(sbase) : "memory")
#define ST16(voff, data, sbase, imm) asm volatile("global_store_dwordx4 %0, %1, %2 offset:" #imm " sc0 sc1\n\ts_nop 1" :: "v"(voff), "v"(data), "s"(sbase) : "memory")
#define WAITV(n) asm volatile("s_waitcnt vmcnt(" #n ")" ::: "memory")
#define PIN8(v) asm volatile("" : "+v"(v[0]), "+v"(v[1]), "+v"(v[2]), "+v"(v[3]), "+v"(v[4]), "+v"(v[5]), "+v"(v[6]), "+v"(v[7]) :: "memory")
#define WAIT8(n, v) do { WAITV(n); PIN8(v); } while (0)
DI void unit_load(const UnitAddr& a, f32x4 (&v)[8], unsigned vx, unsigned vw) {
  const unsigned voff = a.isx ? vx : vw;
  const unsigned s1 = rfl(a.isx ? 16u : N * 4u), s2 = rfl(a.isx ? 16u * KS * 4u : N * 8u);
  const char* src = uptr(a.src);
#pragma unroll
  for (int i = 0; i < 8; ++i) { const char* b = src + (size_t)((i & 1) * s1 + (i >> 1) * s2); LD16(v[i], voff, b); }
}
DI void cvt_w(f32x4 (&v)[8], u32x4 (&o)[4]) {
#pragma unroll
  for (int i = 0; i < 8; ++i) v[i] *= 8.0f * 1.44269504088896f;
#pragma unroll
  for (int c = 0; c < 4; ++c) o[c] = (u32x4){pk16(v[0][c], v[1][c]), pk16(v[2][c], v[3][c]), pk16(v[4][c], v[5][c]), pk16(v[6][c], v[7][c])};
}
DI void cvt_x(f32x4 (&v)[8], u32x4 (&o)[4]) {
#pragma unroll
  for (int i = 0; i < 8; ++i) v[i] *= 0.125f;
#pragma unroll
  for (int f = 0; f < 4; ++f) o[f] = (u32x4){pk16(v[2 * f][0], v[2 * f][1]), pk16(v[2 * f][2], v[2 * f][3]), pk16(v[2 * f + 1][0], v[2 * f + 1][1]), pk16(v[2 * f + 1][2], v[2 * f + 1][3])};
}
constexpr unsigned TOP = 0x40004000u;
constexpr int pos(int i) { return i < 14 ? i : 15; }
DI unsigned hide(unsigned w16, int i) { return (pos(i) < 14 ? w16 << (14 - pos(i)) : w16 >> 1) & TOP; }
DI unsigned seek(unsigned w, int i) { return (pos(i) < 14 ? w >> (14 - pos(i)) : w << 1) & (0x00010001u << pos(i)); }
DI void export_unit(const u32x4 (&o)[4], unsigned slot, unsigned mask, const char* ex, int lane) {
  const unsigned v0 = slot * 4096u + lane * 16;
  ST16(v0, o[0], ex, 0); ST16(v0, o[1], ex, 1024); ST16(v0, o[2], ex, 2048);
  const u32x3 o3 = {o[3][0], o[3][1], o[3][2]};
  asm volatile("global_store_dwordx3 %0, %1, %2 offset:3072 sc0 sc1\n\ts_nop 1" :: "v"(v0 - lane * 4), "v"(o3), "s"(ex) : "memory");
  asm volatile("global_store_dword %0, %1, %2 offset:3840 sc0 sc1" :: "v"(v0 - lane * 12), "v"(o[3][3]), "s"(ex) : "memory");
  asm volatile("s_mov_b64 exec, 1\n\tglobal_store_dword %0, %1, %2 sc0 sc1\n\ts_mov_b64 exec, -1" :: "v"(EX_FLAGS + slot * 4u), "v"(mask), "s"(ex) : "memory");
}
constexpr unsigned TOPF = 0x60000000u;
DI unsigned hidef(unsigned w, int i) { return (w << (29 - 2 * i)) & TOPF; }
DI unsigned seekf(unsigned w, int i) { return (w >> (29 - 2 * i)) & (3u << (2 * i)); }
DI unsigned fscale(unsigned w, float s) { return __builtin_bit_cast(unsigned, __builtin_bit_cast(float, w) * s); }
DI bool export_out(const f32x4 (&y)[4], unsigned slot, const char* ex, int lane) {
  u32x4 o[4]; unsigned odd = 0;
  const u32x4 b[4] = {__builtin_bit_cast(u32x4, y[0]), __builtin_bit_cast(u32x4, y[1]), __builtin_bit_cast(u32x4, y[2]), __builtin_bit_cast(u32x4, y[3])};
#pragma unroll
  for (int i = 0; i < 16; ++i) { const unsigned u = b[i >> 2][i & 3], t = fscale(u, 0x1p-70f);
    o[i >> 2][i & 3] = t; odd |= (t & TOPF) | (fscale(t, 0x1p70f) ^ u); }
  const bool raw = __builtin_amdgcn_ballot_w64(odd != 0) != 0;
  const unsigned w = (o[3][3] & 0x1fffffffu) | ((o[3][3] >> 2) & 0x20000000u);
#pragma unroll
  for (int i = 0; i < 16; ++i) o[i >> 2][i & 3] = raw || i == 15 ? b[i >> 2][i & 3] : o[i >> 2][i & 3] | hidef(w, i);
  export_unit(o, slot, raw ? ~0u : ~TOPF, ex, lane);
  return raw;
}
DI unsigned convert(const Params& p, const char* ws, LAS unsigned char* lds, const char* tell, unsigned x, unsigned u, int wid, int lane) {
  const unsigned vx = (lane & 15) * (KS * 4) + (lane >> 4) * 32, vw = (lane >> 4) * (8u * N * 4u) + (lane & 15) * 16;
  f32x4 v[8]; u32x4 o[4];
  UnitAddr cur = cv_unit(p, u);
  unit_load(cur, v, vx, vw);
  const unsigned call = get_call(ws, lds, x, wid, true);
  const char* ex = ws + CTRL + EX_OFF;
  WAIT8(0, v);
  for (bool first = true, told = false; u < (unsigned)NCV; first = false) {
    if (cur.isx) cvt_x(v, o); else cvt_w(v, o);
    asm volatile("" : "+v"(o[0]), "+v"(o[1]), "+v"(o[2]), "+v"(o[3]));
    if (cur.own) { const unsigned v0 = cur.own + lane * 16, v1 = v0 + 4096u; ST16(v0, o[0], ws, 0); ST16(v0, o[1], ws, 2048); ST16(v1, o[2], ws, 0); ST16(v1, o[3], ws, 2048); }
    const u32x4 all = o[0] | o[1] | o[2] | o[3];
    const unsigned raw = __builtin_amdgcn_ballot_w64(((all[0] | all[1] | all[2] | all[3]) & TOP) != 0) != 0, w16 = raw ? 0u : o[3][3];
#pragma unroll
    for (int i = 0; i < 15; ++i) o[i >> 2][i & 3] |= hide(w16, i);
    export_unit(o, cur.slot, raw ? ~0u : ~TOP, ex, lane);
    if (raw && !told) { told = true; TELL("global_atomic_umax", tell + OFF_PROG + (48 + p.rank) * 16, call); }
    u += 4 * NG;
    if (u < (unsigned)NCV) cur = cv_unit(p, u);
    if (first) { WAITV(0); WG_BAR(); if (wid == 0) TELL("global_atomic_add", tell + OFF_RDY + p.rank * 1024 + x * 128, 1u); }
    unit_load(cur, v, vx, vw);
    WAIT8(0, v);
  }
  WG_BAR();
  if (wid == 0) TELL("global_atomic_add", tell + OFF_RDY + 8192 + p.rank * 1024 + x * 128, 1u);
  return call;
}

struct Cursor { unsigned e, j, sb, size, beg; };
DI void cur_advance(Cursor& c, unsigned n, unsigned size_opt) {
  c.e += n;
  while (c.e >= c.size) {
    c.e -= c.size;
    if (++c.j == 16u) { c.j = 0; ++c.sb;
      c.size = c.sb == 1 ? C_<sb_units(1)> : c.sb == 2 ? C_<sb_units(2)> : c.sb == 3 ? C_<sb_units(3)> : c.sb == 4 ? C_<sb_units(4)> : size_opt;
      c.beg = c.sb == 1 ? C_<sb_beg(1)> : c.sb == 2 ? C_<sb_beg(2)> : c.sb == 3 ? C_<sb_beg(3)> : c.sb == 4 ? C_<sb_beg(4)> : C_<sb_beg(5)>; }
  }
}
struct Pos { unsigned slot, dst, tile; };
DI Pos cur_pos(const Cursor& c, unsigned rank) {
  unsigned e = c.e;
  if (c.sb == 0) {
    if (e < 64u) return Pos{c.j * 576u + e, a_off(e >> 2) + c.j * (unsigned)ACH + (e & 3u) * 8192u, TILEB};
    e -= 64u;
  }
  const unsigned vs = c.beg + e;
  return Pos{c.j * 576u + 64u + real_strip(vs, rank), b_off(vs >> 1) + c.j * (unsigned)BCH + (vs & 1u) * 8192u, HTB};
}
template <int C, class G> DI void pull(G (&g)[NR], unsigned vs, unsigned self, char* const (&base)[NR]) {
#pragma unroll
  for (int d = 0; d < NR - 1; ++d) {
    if constexpr (C < 3) asm volatile("global_load_dwordx4 %0, %1, %2 offset:%3 sc0 sc1" : "+v"(g[d]) : "v"(vs), "s"(base[d]), "i"(C * 1024) : "memory");
    else                 asm volatile("global_load_dwordx3 %0, %1, %2 offset:3072 sc0 sc1" : "+v"(g[d]) : "v"(vs), "s"(base[d]) : "memory");
  }
  if constexpr (C < 3) asm volatile("s_cmp_eq_u32 %4, 0\n\ts_cbranch_scc1 1f\n\tglobal_load_dwordx4 %0, %1, %2 offset:%3 sc0 sc1\n1:" : "+v"(g[7]) : "v"(vs), "s"(base[7]), "i"(C * 1024), "s"(self) : "memory", "scc");
  else                 asm volatile("s_cmp_eq_u32 %3, 0\n\ts_cbranch_scc1 1f\n\tglobal_load_dwordx3 %0, %1, %2 offset:3072 sc0 sc1\n1:" : "+v"(g[7]) : "v"(vs), "s"(base[7]), "s"(self) : "memory", "scc");
}
template <int C, class G> DI void put(const G (&g)[NR], unsigned (&acc)[NR], const unsigned (&keep)[NR], const unsigned (&w16)[NR], const unsigned (&vd)[NR], unsigned self, const char* base) {
#pragma unroll
  for (int d = 0; d < NR; ++d) {
    u32x4 o;
#pragma unroll
    for (int i = 0; i < (C < 3 ? 4 : 3); ++i) { acc[d] = (C + i ? acc[d] : 0u) | seek(g[d][i], C * 4 + i); o[i] = g[d][i] & keep[d]; }
    if constexpr (C == 3) o[3] = keep[d] == ~TOP ? acc[d] : w16[d];
    if (d < NR - 1) asm volatile("global_store_dwordx4 %0, %1, %2 offset:%3 sc0 sc1\n\ts_nop 1" :: "v"(vd[d]), "v"(o), "s"(base), "i"(C * 2048 - 4096) : "memory");
    else asm volatile("s_cmp_eq_u32 %4, 0\n\ts_cbranch_scc1 1f\n\tglobal_store_dwordx4 %0, %1, %2 offset:%3 sc0 sc1\n\ts_nop 1\n1:" :: "v"(vd[d]), "v"(o), "s"(base), "i"(C * 2048 - 4096), "s"(self) : "memory", "scc");
  }
  if constexpr (C < 3) PIN8(acc);
}
template <int C, class G> DI void put_out(const G (&g)[NR], unsigned (&acc)[NR], const unsigned (&keep)[NR], const unsigned (&w16)[NR], const unsigned (&vd)[NR], const char* base) {
#pragma unroll
  for (int d = 0; d < NR - 1; ++d) {
    u32x4 o; const bool raw = keep[d] == ~0u;
#pragma unroll
    for (int i = 0; i < (C < 3 ? 4 : 3); ++i) { acc[d] = (C + i ? acc[d] : 0u) | seekf(g[d][i], C * 4 + i); o[i] = raw ? g[d][i] : fscale(g[d][i] & ~TOPF, 0x1p70f); }
    if constexpr (C == 3) o[3] = raw ? w16[d] : fscale((acc[d] & 0x1fffffffu) | ((acc[d] << 2) & 0x80000000u), 0x1p70f);
    asm volatile("global_store_dwordx4 %0, %1, %2 nt\n\ts_nop 1" :: "v"(vd[d]), "v"(o), "s"(base + (size_t)C * (N * 4)) : "memory");
  }
  if constexpr (C < 3) PIN8(acc);
}
#define UNDEF8(g) asm volatile("" : "=v"(g[0]), "=v"(g[1]), "=v"(g[2]), "=v"(g[3]), "=v"(g[4]), "=v"(g[5]), "=v"(g[6]), "=v"(g[7]))
DI void comm_role(const Params& p, LAS unsigned char* lds, unsigned bid, unsigned x) {
  const int tid = threadIdx.x, wid = rfl(tid >> 6), lane = tid & 63;
  const unsigned gw = bid * 4 + wid, rank = rfl(p.rank);
  char* const ws = upick(p.ws, rank);
  const unsigned call = get_call(ws, lds, x, wid, false);
  LAS unsigned* tkw = (LAS unsigned*)lds;
  if (wid == 0) queue_open(ws, OFF_BATQ, call);
  char* peer[NR]; unsigned pid[NR];
#pragma unroll
  for (int d = 0; d < NR; ++d) { pid[d] = d < 7 ? (rank + 1u + (d + gw) % 7u) & 7u : rank; peer[d] = upick(p.ws, pid[d]) + CTRL + EX_OFF; }
  bool raw, bad = !wait_ready(ws, call, 0, raw), known = false; unsigned open = 0, level = 0, nb = NB, ne = NB + 1;
  {
    unsigned t0 = bid, t1 = t0 + NCOMM, t2 = t1 + NCOMM, pub = 0, tk = 0;
    const unsigned qbase = (call << QSH) - 3u * NCOMM, one = 1u, qo = OFF_BATQ, po = OFF_PROG + bid * 16;
    const char* mine = ws + CTRL + 4096; const char* out = upick(p.out, rank);
    auto publish = [&](unsigned inc) {
      WG_BAR(); if (wid == 1 && inc) asm volatile("s_mov_b64 exec, 1\n\tglobal_atomic_add %0, %1, %2 sc1\n\ts_mov_b64 exec, -1" :: "v"(po), "v"(inc), "s"(ws) : "memory"); };
    Cursor c = {0u, 0u, 0u, (unsigned)C_<sb_units(0)>, 0u};
    cur_advance(c, gw, 0);
    u32x4 ga[NR], gb[NR]; u32x3 gc[NR]; UNDEF8(ga); UNDEF8(gb); UNDEF8(gc);
    unsigned acc[NR] = {}, keep[NR], w16[NR], vd[NR], self = 0;
    Pos q = cur_pos(c, rank);
    pull<0>(ga, q.slot * 4096u + lane * 16u, 0u, peer);
    WAITV(0);
    for (unsigned r = 0; t0 < ne; ++r) {
      if (r > 0 && wid == 0) *(volatile LAS unsigned*)tkw = rfl(tk) - qbase;
      asm volatile("s_cmp_lg_u32 %4, 0\n\ts_cbranch_scc1 1f\n\ts_mov_b64 exec, 1\n\tglobal_atomic_add %0, %1, %2, %3 sc0\n\ts_mov_b64 exec, -1\n1:\n\ts_waitcnt lgkmcnt(0)"
                   : "+v"(tk) : "v"(qo), "v"(one), "s"(ws), "s"(wid) : "memory", "scc");
      const unsigned vs = q.slot * 4096u + lane * 16u; const bool fin = t0 >= nb;
#define PUT(C, g) do { if (fin) put_out<C>(g, acc, keep, w16, vd, out); else put<C>(g, acc, keep, w16, vd, self, mine); } while (0)
#pragma unroll
      for (int d = 0; d < NR; ++d) { keep[d] = ~TOP; w16[d] = 0;
        vd[d] = q.dst + (fin ? pid[d] * (OWNS * 256u) + (lane >> 4) * (4u * N * 4u) + (lane & 15) * 16u : (pid[d] >> 1) * q.tile + (pid[d] & 1u) * 1024u + lane * 16u); }
      if (raw) {
#pragma unroll
        for (int d = 0; d < NR; ++d)
          asm volatile("global_load_dword %0, %2, %4 sc0 sc1\n\tglobal_load_dword %1, %3, %4 offset:3840 sc0 sc1"
                       : "=&v"(keep[d]), "=&v"(w16[d]) : "v"(EX_FLAGS + q.slot * 4u), "v"(vs - lane * 12u), "s"(peer[d]) : "memory");
        WAIT8(0, keep); PIN8(w16);
      }
      pull<1>(gb, vs, self, peer); WAIT8(14, ga); PUT(0, ga);
      pull<2>(ga, vs, self, peer); WAIT8(14, gb);
      if (r > 0) { const unsigned now = fin ? NB : t0; publish(now - pub); pub = now; }
      PUT(1, gb);
      pull<3>(gc, vs - lane * 4u, self, peer); WAIT8(14, ga); PUT(2, ga);
      if (t1 >= (unsigned)NBO && !known) { known = true; if (!wait_level(ws, call, level)) bad = true; nb = NB - 28 * level; ne = nb + 64 * (OWNF + level); }
      if (t1 < nb) cur_advance(c, (t1 - t0) * 4u, 7 * (NOPT - level));
      { const unsigned must = t1 >= nb ? 4 : c.sb >= 4 ? 3 : c.sb >= 3 ? 2 : (c.sb > 0 || c.j >= (unsigned)J1) ? 1 : 0;
        while (open < must) { ++open; if (!((open & 2) ? wait_words(ws, call, OFF_DONE + open * 4, 0, 1ull, call * NG) : wait_ready(ws, call, open == 1 ? 1 : 2, raw))) bad = true; } }
      unsigned sn = 0;
      if (t1 < nb) { q = cur_pos(c, rank); sn = c.sb - 3u < 2u ? c.sb : 0u; }
      else if (t1 < ne) { const unsigned f = (t1 - nb) * 4 + wid, nf = OWNF + level, rg = f / nf, i = f - rg * nf;
        q = Pos{EXO + rg * OWNS + i, rg * (16u * N * 4u) + (NREP + i) * 256u, 0u}; }
      pull<0>(ga, q.slot * 4096u + lane * 16u, sn, peer); WAIT8(14, gc); PUT(3, gc);
      self = sn;
      t0 = t1; t1 = t2;
      if (r > 0) t1 = rfl(*(volatile LAS unsigned*)tkw);
      if (t1 <= t0) t1 = ~0u;
      asm volatile("" : "+v"(tk));
      tkw = (LAS unsigned*)lds + ((r & 1u) ^ 1u);
    }
    WAIT8(0, ga);
    publish(NB - pub);
    if (wid == 0) TELL("global_atomic_add", pick(p.ws, lane & 7) + OFF_PROG + (40 + rank) * 16, 1u);
    if (rfl(__hip_atomic_load((GAS unsigned*)(ws + OFF_TMO), __ATOMIC_RELAXED, __HIP_MEMORY_SCOPE_AGENT)) == call) bad = true;
    if (bad) *(volatile GAS float*)(out + (size_t)(NREP + OWNS * rank) * 256) = __builtin_nanf("");
  }
  asm volatile(".Lgk_code_end:");
}

typedef f16x8 FA[4][2];
typedef f16x8 FB[2][2];
#define DSR(DST, ADDR, OFF) asm volatile("ds_read_b128 %0, %1 offset:%2" : "+v"(DST) : "v"(ADDR), "i"(OFF))
template <bool INV, bool ZERO, int CB, bool RDA, int AOFF, bool RDB, int BOFF, bool GL, int WA, int PRE = 0, int VM = 0> struct Sub {
  static constexpr bool inv = INV, zero = ZERO, rda = RDA, rdb = RDB, gl = GL; static constexpr int cb = CB, aoff = AOFF, boff = BOFF, wa = WA, pre = PRE, vm = VM; };
template <int I, class S> DI void sstep(f32x4 (&acc)[4][4], FA& A, FB& B, FA& RA, unsigned ra, FB& RB, unsigned rb, unsigned voff, const char* gsrc, unsigned m0v) {
  constexpr int m = I >> 1, n = (m & 1) ? 1 - (I & 1) : (I & 1), k0 = I & 1, k1 = 1 - k0;
  if constexpr (I == 0 && S::pre == 1) asm volatile("s_waitcnt lgkmcnt(0)" ::: "memory");
  if constexpr (I == 0 && S::pre == 2) asm volatile("s_waitcnt vmcnt(%0) lgkmcnt(0)\n\ts_barrier" :: "i"(S::vm) : "memory");
  if constexpr (S::wa != 0 && (I & 1) == 0) {
    constexpr int later = (6 - 2 * m) + (S::wa == 1 ? I : (I < 4 ? I : 4));
    asm volatile("s_waitcnt lgkmcnt(%0)" :: "i"(later));
  }
#define MM2(CON, C0) do { if constexpr (S::gl && (I & 1)) \
    asm volatile("v_mfma_f32_16x16x32_f16 %0, %1, %2, " C0 "\n\tglobal_load_lds_dwordx4 %5, %6 offset:%7 sc0 sc1\n\tv_mfma_f32_16x16x32_f16 %0, %3, %4, %0" \
      : CON(acc[m][S::cb + n]) : "v"(A[m][k0]), "v"(B[n][k0]), "v"(A[m][k1]), "v"(B[n][k1]), "v"(voff), "s"(gsrc), "i"((I >> 1) * 1024) : "memory"); \
    else asm volatile("v_mfma_f32_16x16x32_f16 %0, %1, %2, " C0 "\n\tv_mfma_f32_16x16x32_f16 %0, %3, %4, %0" \
      : CON(acc[m][S::cb + n]) : "v"(A[m][k0]), "v"(B[n][k0]), "v"(A[m][k1]), "v"(B[n][k1])); } while (0)
  if constexpr (S::zero) { if constexpr (S::inv) MM2("=&v", "0"); else MM2("=&a", "0"); }
  else                   { if constexpr (S::inv) MM2("+v", "%0"); else MM2("+a", "%0"); }
#undef MM2
  if constexpr (S::rda) DSR(RA[I >> 1][I & 1], ra, S::aoff + I * 1024);
  if constexpr (S::rdb && I < 4) DSR(RB[I >> 1][I & 1], rb, S::boff + I * 1024);
  if constexpr (S::gl && I == 0) asm volatile("s_mov_b32 m0, %0" :: "s"(m0v));
}
template <class S, int... Is> DI void sub_(f32x4 (&acc)[4][4], FA& A, FB& B, FA& RA, unsigned ra, FB& RB, unsigned rb, unsigned voff, const char* gsrc, unsigned m0v, std::integer_sequence<int, Is...>) {
  (sstep<Is, S>(acc, A, B, RA, ra, RB, rb, voff, gsrc, m0v), ...);
}
template <class S> DI void sub(f32x4 (&acc)[4][4], FA& A, FB& B, FA& RA, unsigned ra, FB& RB, unsigned rb, unsigned voff, const char* gsrc, unsigned m0v) {
  sub_<S>(acc, A, B, RA, ra, RB, rb, voff, gsrc, m0v, std::make_integer_sequence<int, 8>{});
}
DI void dma4(unsigned voff, const char* g, unsigned m0v) {
  asm volatile("s_mov_b32 m0, %2\n\ts_nop 0\n\tglobal_load_lds_dwordx4 %0, %1 sc0 sc1\n\tglobal_load_lds_dwordx4 %0, %1 offset:1024 sc0 sc1\n\t"
               "global_load_lds_dwordx4 %0, %1 offset:2048 sc0 sc1\n\tglobal_load_lds_dwordx4 %0, %1 offset:3072 sc0 sc1" :: "v"(voff), "s"(g), "s"(m0v) : "memory");
}
struct Lane { unsigned voff, m0w, rA, rB0; const char* line; };
struct Walk { const char *a, *b, *an, *bn; unsigned oa, ob; };
DI void walk(const char*& next, unsigned& o, const char* base, unsigned mask) {
  asm volatile("s_sub_u32 %0, %0, %1\n\ts_and_b32 %0, %0, %1" : "+s"(o) : "s"(mask) : "scc");
  next = base + o;
  asm volatile("" : "+s"(next));
}
template <int NQ, int P, bool ZERO, int TAIL>
DI void kstep(f32x4 (&acc)[NQ][2][4][4], FA& A0, FA& A1, FB& Ba, FB& Bb, const Lane& L, unsigned rB1, const char* a2, const char* b2, const char* bl1, Walk& W) {
  constexpr int AS = P * TILEB, ASN = (P ^ 1) * TILEB, BW = 65536 + P * NQ * HTB, BWN = 65536 + (P ^ 1) * NQ * HTB, H = HTB;
  constexpr bool G = TAIL == 0, G1 = TAIL < 2, Z = ZERO;
  const unsigned rBc = P ? rB1 : L.rB0, rBn = P ? L.rB0 : rB1, rA = L.rA, vo = L.voff, mw = L.m0w;
  sub<Sub<false, Z, 0, true, AS + H, false, 0, false, 1>>(acc[0][0], A0, Ba, A1, rA, Bb, rBc, vo, nullptr, 0);
  sub<Sub<false, Z, 0, false, 0, true, 4096, G1, 2>>(acc[0][1], A1, Ba, A0, rA, Bb, rBc, vo, bl1, mw + BWN + (NQ - 1) * H);
  if constexpr (NQ == 3) {
    sub<Sub<false, Z, 2, false, 0, true, H, G, 0, 2, TAIL == 2 ? 4 : 24>>(acc[0][0], A0, Bb, A1, rA, Ba, rBc, vo, a2, mw + AS);
    sub<Sub<false, Z, 2, false, 0, false, 0, G, 0>>(acc[0][1], A1, Bb, A0, rA, Ba, rBc, vo, b2, mw + BW);
    sub<Sub<false, Z, 0, false, 0, true, H + 4096, false, 0, 1>>(acc[1][0], A0, Ba, A1, rA, Bb, rBc, vo, nullptr, 0);
    if constexpr (G) { if constexpr (P == 0) walk(W.an, W.oa, W.a, 15 * ACH + 2 * TILEB); else walk(W.bn, W.ob, W.b, 15 * BCH + 2 * HTB); }
    sub<Sub<false, Z, 0, false, 0, false, 0, false, 0>>(acc[1][1], A1, Ba, A0, rA, Bb, rBc, vo, nullptr, 0);
    sub<Sub<false, Z, 2, false, 0, true, 2 * H, G, 0, 2, TAIL == 2 ? 0 : TAIL == 1 ? 20 : 28>>(acc[1][0], A0, Bb, A1, rA, Ba, rBc, vo, a2 + H, mw + AS + H);
    sub<Sub<false, Z, 2, false, 0, false, 0, G, 0>>(acc[1][1], A1, Bb, A0, rA, Ba, rBc, vo, b2 + BPANEL, mw + BW + H);
    sub<Sub<true, Z, 0, false, 0, true, 2 * H + 4096, false, 0, 1>>(acc[2][0], A0, Ba, A1, rA, Bb, rBc, vo, nullptr, 0);
    sub<Sub<true, Z, 0, false, 0, false, 0, false, 0>>(acc[2][1], A1, Ba, A0, rA, Bb, rBc, vo, nullptr, 0);
    sub<Sub<true, Z, 2, false, 0, G1, 0, false, 0, 2, TAIL == 2 ? 0 : TAIL == 1 ? 8 : 24>>(acc[2][0], A0, Bb, A1, rA, Ba, rBn, vo, nullptr, 0);
    sub<Sub<true, Z, 2, G1, ASN, false, 0, false, 0>>(acc[2][1], A1, Bb, A0, rA, Ba, rBn, vo, nullptr, 0);
  } else {
    sub<Sub<false, Z, 2, false, 0, true, H, G, 0, 2, TAIL == 2 ? 0 : 16>>(acc[0][0], A0, Bb, A1, rA, Ba, rBc, vo, a2, mw + AS);
    sub<Sub<false, Z, 2, false, 0, false, 0, G, 0>>(acc[0][1], A1, Bb, A0, rA, Ba, rBc, vo, a2 + H, mw + AS + H);
    sub<Sub<false, Z, 0, false, 0, true, H + 4096, G, 0, 1>>(acc[1][0], A0, Ba, A1, rA, Bb, rBc, vo, b2, mw + BW);
    if constexpr (G) { if constexpr (P == 0) walk(W.an, W.oa, W.a, 15 * ACH + 2 * TILEB); else walk(W.bn, W.ob, W.b, 15 * BCH + 2 * HTB); }
    sub<Sub<false, Z, 0, false, 0, false, 0, false, 0>>(acc[1][1], A1, Ba, A0, rA, Bb, rBc, vo, nullptr, 0);
    sub<Sub<false, Z, 2, false, 0, G1, 0, false, 0, 2, TAIL == 2 ? 0 : TAIL == 1 ? 4 : 16>>(acc[1][0], A0, Bb, A1, rA, Ba, rBn, vo, nullptr, 0);
    sub<Sub<false, Z, 2, G1, ASN, false, 0, false, 0>>(acc[1][1], A1, Bb, A0, rA, Ba, rBn, vo, nullptr, 0);
  }
}
struct Tile { int pm, vp, nq, k0 = 0; };
DI Tile tile_y(unsigned band, unsigned s) { const unsigned b = s >> 3;
  return Tile{(int)(band * 8 + (s & 7u)), (int)(3 * (b < (unsigned)B0 ? (b * 3u) % B0 : b)), 3}; }
DI void split_band(unsigned t, unsigned n, unsigned& band, unsigned& s) {
  const unsigned full = n & ~31u;
  if (t < 2 * full) { band = (t >> 5) & 1u; s = (t >> 6) * 32u + (t & 31u); }
  else { t -= 2 * full; band = t >= n - full; s = full + t - band * (n - full); }
}
template <int L> constexpr int BYT = by(L < NOPT ? L : NOPT) - BYS;
DI Tile tile_dyn(unsigned t, unsigned level) {
  constexpr unsigned TAB = BYT<0> | BYT<1> << 4 | BYT<2> << 8 | BYT<3> << 12 | BYT<4> << 16 | BYT<5> << 20 | BYT<6> << 24 | (unsigned)BYT<7> << 28;
  const unsigned b = (TAB >> (4 * level)) & 15u, vx = 3 * (BYS + b), ry = b * 8, nx = ((VO + 7 * (NOPT - level) + 1) / 2 - vx) / 2 * 8;
  const unsigned lead = ry + nx > (unsigned)HB ? ry + nx - HB : 0u;
  unsigned band, s;
  if (t - 2 * lead < 2 * ry) { split_band(t - 2 * lead, ry, band, s); return tile_y(band, NST * HB + s); }
  if (t >= 2 * lead) t -= 2 * ry;
  if (t >= 2 * nx) return Tile{0, 0, 0};
  split_band(t, nx, band, s);
  return Tile{(int)(band * 8 + (s & 7u)), (int)(vx + 2 * (s >> 3)), 2};
}
struct Need { unsigned u0, du; };
DI Need need_of(const Tile& t, unsigned level) {
  const int end = 2 * (t.vp + t.nq);
  if (end > VO) return Need{4 * NBO + 7 * (NOPT - level), 7 * (NOPT - level)};
  unsigned base = 64, size = C_<sb_units(0)>;
#define SBSEL(i) if (end > C_<sb_beg(i)>) { base = C_<sb_base(i) - sb_beg(i)>; size = C_<sb_units(i)>; }
  SBSEL(1) SBSEL(2) SBSEL(3) SBSEL(4)
#undef SBSEL
  return Need{base + end, size};
}
DI unsigned need_batches(const Need& n, int j) { return (n.u0 + n.du * j + 3) >> 2; }
struct Next { const char* a; const char* b; int nq; };
template <int I> DI void pro_piece(const Next& n, const Lane& L) {
  constexpr unsigned H = HTB, T = TILEB; constexpr size_t P = BPANEL;
  constexpr bool isa3[9] = {1, 0, 1, 0, 0, 1, 0, 1, 0}, isa2[9] = {1, 1, 0, 0, 1, 1, 0, 0, 0};
  constexpr size_t src3[9] = {0, 0, H, P, 2 * P, T, H, T + H, P + H}, src2[9] = {0, H, 0, P, T, T + H, H, 0, 0};
  constexpr unsigned dst3[9] = {0, 4 * H, H, 5 * H, 6 * H, T, 7 * H, T + H, 8 * H}, dst2[9] = {0, H, 4 * H, 5 * H, T, T + H, 6 * H, 0, 0};
  if (n.nq == 3) dma4(L.voff, (isa3[I] ? n.a : n.b) + src3[I], L.m0w + dst3[I]);
  else if (I < 7 && n.nq == 2) dma4(L.voff, (isa2[I] ? n.a : n.b) + src2[I], L.m0w + dst2[I]);
}
template <int... Is> DI void prologue_(const Next& n, const Lane& L, std::integer_sequence<int, Is...>) { (pro_piece<Is>(n, L), ...); }
DI Next next_of(const Tile& t, const char* XA) { return Next{uptr(XA + a_off(t.pm) + koff_a(t.k0)), uptr(XA + b_off(t.vp) + koff_b(t.k0)), t.nq}; }
DI void prologue(const Tile& t, const char* XA, const Lane& L) { prologue_(next_of(t, XA), L, std::make_integer_sequence<int, 9>{}); }
struct Sync { const char* ws; unsigned call; bool all, bad; unsigned level, tk, junk; unsigned pw; };
template <int NQ> DI void kloop(f32x4 (&acc)[NQ][2][4][4], const Tile& t, const char* XA, const Lane& L, Sync& S, unsigned draw, bool pf) {
  const char* cA = uptr(XA + a_off(t.pm)); const char* cB = uptr(XA + b_off(t.vp));
  const unsigned rB1 = L.rB0 + NQ * HTB;
  FA A0, A1; FB Ba, Bb;
  asm volatile("s_waitcnt vmcnt(%0)\n\ts_barrier" :: "i"(NQ == 3 ? 24 : 16) : "memory");
#pragma unroll
  for (int r = 0; r < 4; ++r) asm volatile("ds_read_b128 %0, %1 offset:%2" : "=v"(Ba[r >> 1][r & 1]) : "v"(L.rB0), "i"(r * 1024));
#pragma unroll
  for (int r = 0; r < 8; ++r) asm volatile("ds_read_b128 %0, %1 offset:%2" : "=v"(A0[r >> 1][r & 1]) : "v"(L.rA), "i"(r * 1024));
  asm volatile("" : "=v"(A1[0][0]), "=v"(A1[0][1]), "=v"(A1[1][0]), "=v"(A1[1][1]), "=v"(A1[2][0]), "=v"(A1[2][1]), "=v"(A1[3][0]), "=v"(A1[3][1]),
               "=v"(Bb[0][0]), "=v"(Bb[0][1]), "=v"(Bb[1][0]), "=v"(Bb[1][1]));
  constexpr size_t LP = (size_t)(NQ - 1) * BPANEL;
  const int k0 = t.k0;
  Walk W = {cA, cB, nullptr, nullptr, koff_a((k0 + 2) & 63), koff_b((k0 + 2) & 63)};
  const char* b0 = cB + W.ob;
  { const char* a2 = cA + W.oa;
    kstep<NQ, 0, true, 0>(acc, A0, A1, Ba, Bb, L, rB1, a2, b0, cB + koff_b(k0 + 1) + LP, W);
    kstep<NQ, 1, false, 0>(acc, A0, A1, Ba, Bb, L, rB1, a2 + TILEB, b0 + HTB, b0 + LP, W); }
  const char* pbase = S.ws + OFF_PROG - rfl(L.voff);
  const Need nd = need_of(t, S.level);
  int ev = (S.all || k0 >= 58) ? 60 : 2 + (k0 & 2);
  for (int k = 2; k < 62;) {
    if (k == ev) {
      const int check = k == 60 ? 0 : (k + k0) & 2;
      if (check) {
        unsigned mn; asm volatile("v_mov_b32 %0, %1" : "=v"(mn) : "v"(S.pw));
        int c = (k + k0 + 2) >> 2;
        if (!prog_ge(mn, S.call, need_batches(nd, c))) {
          asm volatile("s_waitcnt lgkmcnt(0)" ::: "memory");
          if (!wait_progress(S.ws, S.call, need_batches(nd, c))) { S.bad = true; S.all = true; c = 15; }
        }
        while (c < 15 && prog_ge(mn, S.call, need_batches(nd, c + 1))) ++c;
        ev = c == 15 ? 60 : 4 * c - k0;
      } else ev = k + 2;
      if (k == 60 && pf) code_prefetch(S.junk, L.voff);
      asm volatile("s_cmp_lg_u32 %3, 0\n\ts_cbranch_scc1 1f\n\tglobal_load_dword %0, %1, %2 sc0 sc1\n1:"
                   : "+v"(S.pw) : "v"(L.voff), "s"(pbase), "s"(check) : "memory", "scc");
      asm volatile("s_cmp_lg_u32 %3, 60\n\ts_cbranch_scc1 1f\n\tv_mov_b32 %0, 1\n\ts_mov_b64 exec, 1\n\tglobal_atomic_add %0, %1, %0, %2 sc0\n\ts_mov_b64 exec, -1\n1:"
                   : "+v"(S.tk) : "v"(L.voff), "s"(S.ws + draw), "s"(draw ? k : 0) : "memory", "scc");
    }
    do {
      const char* a2 = W.an; const char* b2 = W.bn;
      kstep<NQ, 0, false, 0>(acc, A0, A1, Ba, Bb, L, rB1, a2, b2, b0 + HTB + LP, W);
      kstep<NQ, 1, false, 0>(acc, A0, A1, Ba, Bb, L, rB1, a2 + TILEB, b2 + HTB, b2 + LP, W);
      b0 = b2; k += 2;
    } while (k < ev);
  }
  kstep<NQ, 0, false, 1>(acc, A0, A1, Ba, Bb, L, rB1, nullptr, nullptr, b0 + HTB + LP, W);
  kstep<NQ, 1, false, 2>(acc, A0, A1, Ba, Bb, L, rB1, nullptr, nullptr, nullptr, W);
  asm volatile("s_nop 7\n\ts_nop 7" : "+v"(S.tk), "+v"(S.junk), "+v"(S.pw) :: "memory");
}
#define EST(r) do { if constexpr (PREV) asm volatile("global_store_dwordx4 %0, %1, %2 nt" :: "v"(so), "v"(po[r]), "s"(pb + (size_t)(r) * (N * 4)) : "memory"); } while (0)
template <bool INV, bool PREV> DI void epi_block(f32x4 (&a)[4], f32x4 (&o)[4], const f32x4 (&po)[4], const char* pb, unsigned so) {
  const float LO = -86.5617f; const f32x2 C2 = {1.44269504088896f, 1.44269504088896f};
  f32x4 ee[4]; f32x2 d[8];
  if constexpr (!INV) {
#pragma unroll
    for (int r = 0; r < 4; ++r)
      asm volatile("v_accvgpr_read_b32 %0, %4\n\tv_accvgpr_read_b32 %1, %5\n\tv_accvgpr_read_b32 %2, %6\n\tv_accvgpr_read_b32 %3, %7"
                   : "=v"(o[r][0]), "=v"(o[r][1]), "=v"(o[r][2]), "=v"(o[r][3]) : "a"(a[0][r]), "a"(a[1][r]), "a"(a[2][r]), "a"(a[3][r]));
  }
  EST(0);
#pragma unroll
  for (int i = 0; i < 4; ++i) { const f32x4& y = INV ? a[i] : o[i];
    asm volatile("v_max_f32 %0, %8, %4\n\tv_max_f32 %1, %8, %5\n\tv_max_f32 %2, %8, %6\n\tv_max_f32 %3, %8, %7"
                 : "=&v"(ee[i][0]), "=&v"(ee[i][1]), "=&v"(ee[i][2]), "=&v"(ee[i][3]) : "v"(y[0]), "v"(y[1]), "v"(y[2]), "v"(y[3]), "s"(LO)); }
#pragma unroll
  for (int i = 0; i < 4; ++i) {
    asm volatile("v_exp_f32 %0, -%0\n\tv_exp_f32 %1, -%1\n\tv_exp_f32 %2, -%2\n\tv_exp_f32 %3, -%3" : "+v"(ee[i][0]), "+v"(ee[i][1]), "+v"(ee[i][2]), "+v"(ee[i][3]));
    if (i == 1) EST(1);
  }
#pragma unroll
  for (int i = 0; i < 4; ++i) { d[2 * i] = (f32x2){ee[i][0], ee[i][1]}; d[2 * i + 1] = (f32x2){ee[i][2], ee[i][3]}; }
#pragma unroll
  for (int i = 0; i < 8; i += 4)
    asm volatile("v_pk_fma_f32 %0, %0, %4, %4\n\tv_pk_fma_f32 %1, %1, %4, %4\n\tv_pk_fma_f32 %2, %2, %4, %4\n\tv_pk_fma_f32 %3, %3, %4, %4" : "+v"(d[i]), "+v"(d[i + 1]), "+v"(d[i + 2]), "+v"(d[i + 3]) : "s"(C2));
  EST(2);
  if constexpr (!INV) {
#pragma unroll
    for (int i = 0; i < 8; i += 2) {
      asm volatile("v_rcp_f32 %0, %0\n\tv_rcp_f32 %1, %1\n\tv_rcp_f32 %2, %2\n\tv_rcp_f32 %3, %3" : "+v"(d[i][0]), "+v"(d[i][1]), "+v"(d[i + 1][0]), "+v"(d[i + 1][1]));
      if (i == 4) EST(3);
    }
#pragma unroll
    for (int r = 0; r < 4; ++r) {
      f32x2 lo = {o[r][0], o[r][1]}, hi = {o[r][2], o[r][3]};
      asm volatile("v_pk_mul_f32 %0, %0, %2\n\tv_pk_mul_f32 %1, %1, %3" : "+v"(lo), "+v"(hi) : "v"(d[2 * r]), "v"(d[2 * r + 1]));
      o[r] = (f32x4){lo[0], lo[1], hi[0], hi[1]};
    }
  } else {
#pragma unroll
    for (int r = 0; r < 4; ++r) {
      asm volatile("v_rcp_f32 %0, %4\n\tv_rcp_f32 %1, %5\n\tv_rcp_f32 %2, %6\n\tv_rcp_f32 %3, %7" : "=&v"(o[r][0]), "=&v"(o[r][1]), "=&v"(o[r][2]), "=&v"(o[r][3])
                   : "v"(d[r >> 1][r & 1]), "v"(d[2 + (r >> 1)][r & 1]), "v"(d[4 + (r >> 1)][r & 1]), "v"(d[6 + (r >> 1)][r & 1]));
      if (r == 2) EST(3);
    }
#pragma unroll
    for (int r = 0; r < 4; ++r)
      asm volatile("v_mul_f32 %0, %4, %0\n\tv_mul_f32 %1, %5, %1\n\tv_mul_f32 %2, %6, %2\n\tv_mul_f32 %3, %7, %3"
                   : "+v"(o[r][0]), "+v"(o[r][1]), "+v"(o[r][2]), "+v"(o[r][3]) : "v"(a[0][r]), "v"(a[1][r]), "v"(a[2][r]), "v"(a[3][r]));
  }
  if constexpr (PREV) asm volatile("" :: "v"(po[0]), "v"(po[1]), "v"(po[2]), "v"(po[3]));
}
struct Own { const char* ex; unsigned strip, slot; int lane; bool raw; };
template <int NQ, int I> DI void epi_one(f32x4 (&acc)[NQ][2][4][4], f32x4 (&oo)[2][4], const char* (&base)[3], unsigned so, const Next& n, const Lane& L, Own& own) {
  constexpr int q = NQ == 3 ? (I / 8 + 2) % 3 : I / 8, J = I > 0 ? I - 1 : 0, pq = NQ == 3 ? (J / 8 + 2) % 3 : J / 8;
  if constexpr (I < 9) pro_piece<I>(n, L);
  epi_block<q == 2, (I > 0)>(acc[q][(I >> 2) & 1][I & 3], oo[I & 1], oo[J & 1], base[pq] + (size_t)(((J >> 2) & 1) * 128 + (J & 3) * 16) * (N * 4), so);
  if constexpr (NQ == 3) { if (__builtin_expect(own.strip + 2 * q < (unsigned)OWNS, 0)) own.raw |= export_out(oo[I & 1], own.slot + (((I >> 2) & 1) * 8 + (I & 3)) * OWNS + 2 * q, own.ex, own.lane); }
  if constexpr (I == 8 * NQ - 1) {
    constexpr bool PREV = true; const f32x4 (&po)[4] = oo[I & 1]; const char* pb = base[q] + (size_t)(128 + 48) * (N * 4);
    EST(0); EST(1); EST(2); EST(3);
    asm volatile("s_nop 1" :: "v"(po[0]), "v"(po[1]), "v"(po[2]), "v"(po[3]));
  }
}
template <int NQ, int... Is> DI void epilogue(f32x4 (&acc)[NQ][2][4][4], const char* (&base)[3], unsigned so, const Next& n, const Lane& L, Own& own, std::integer_sequence<int, Is...>) {
  f32x4 oo[2][4];
  (epi_one<NQ, Is>(acc, oo, base, so, n, L, own), ...);
}
struct Who { unsigned band, s, r; };
template <int NQ> DI Tile run_tile(const Params& p, LAS unsigned char* lds, const Tile t, const char* XA, const Lane& L, Sync& S, int ui, int wid, const Who& w) {
  const unsigned call = S.call;
  f32x4 acc[NQ][2][4][4];
  const bool dyn = ui >= NST - 1;
  kloop<NQ>(acc, t, XA, L, S, wid ? 0u : dyn ? OFF_TILEQ : ui == NST - 2 ? OFF_PACE : 0u, ui == 0 || ui >= NST - 1);
  if (NQ == 3 && ui == NST - 2 && wid == 0 && rfl(S.tk) - (call << QSH) == NG / 2) {
    const unsigned mn = S.pw; unsigned l = 0;
    if (prog_ge(mn, call, TH0)) { l = 2; while (l < (unsigned)NOPT && prog_ge(mn, call, TH2 + THD * (l - 2))) ++l; }
    asm volatile("s_mov_b64 exec, 1\n\tglobal_atomic_umax %0, %1, %2 sc1\n\ts_mov_b64 exec, -1" :: "v"(OFF_PROG + 32 * 16), "v"(call * 8 + l), "s"(S.ws) : "memory");
  }
  if (NQ == 3 && ui == 0) {
    const unsigned long long t1 = rtime() + w.r * STAGGER;
    while (rtime() < t1) __builtin_amdgcn_s_sleep(4);
    asm volatile("global_load_dword %0, %1, %2 sc0 sc1\n\ts_waitcnt vmcnt(0)" : "+v"(S.pw) : "v"(L.voff), "s"(uptr(S.ws + OFF_PROG - rfl(L.voff))) : "memory");
  }
  const int lane = lane_id();
  Tile nt;
  if (dyn) {
    if (wid == 0) *(volatile LAS unsigned*)(lds + LDS_WORD) = rfl(S.tk);
    asm volatile("s_waitcnt lgkmcnt(0)" ::: "memory");
    WG_BAR();
    const unsigned tk = rfl(*(volatile LAS unsigned*)(lds + LDS_WORD));
    if (all_ge(S.pw, L_LEVEL, call * 8)) S.level = level_of(S.pw);
    else if (!wait_level(S.ws, call, S.level)) S.bad = true;
    nt = tile_dyn(tk - (call << QSH), S.level);
  } else nt = tile_y(w.band, (ui + 1) * HB + w.s);
  nt.k0 = NQ == nt.nq && ui > 0 ? (t.k0 + SKIP) & 63 : nt.nq == 3 ? ((w.r * STAGGER + TSTEP) / (2 * TSTEP)) * 2 : ((w.r * STAGGER * 3 + 2 * TSTEP) / (4 * TSTEP)) * 2;
  nt.pm = rfl(nt.pm); nt.vp = rfl(nt.vp); nt.nq = rfl(nt.nq); nt.k0 = rfl(nt.k0);
  bool early = false;
  if (nt.nq) {
    const unsigned mn = S.pw;
    if (prog_ge(mn, call, NB)) S.all = true;
    early = S.all || prog_ge(mn, call, need_batches(need_of(nt, S.level), min((nt.k0 + 3) >> 2, 15)));
  }
  Next nx = next_of(nt, XA); if (!early) nx.nq = 0;
  {
    const unsigned so = (unsigned)(4 * (lane >> 4)) * (N * 4) + (lane & 15) * 16;
    const size_t ro = (size_t)(t.pm * 256 + (wid >> 1) * 64) * (N * 4);
    const char* rows = upick(p.out, p.rank) + ro;
    auto col = [&](int vp) { const unsigned vs = 2 * vp + (wid & 1);
      return vs >= VO + 7 * (NOPT - S.level) ? S.ws + CTRL + BUF_BYTES + ro : rows + (size_t)real_strip(vs, p.rank) * 256; };
    const char* base[3] = {uptr(col(t.vp)), uptr(col(t.vp + 1)), uptr(col(t.vp + 2))};
    const unsigned s0 = 2 * t.vp + (wid & 1);
    Own own = {S.ws + CTRL + EX_OFF, s0, EXO + (t.pm * 16 + (wid >> 1) * 4) * OWNS + s0, lane, false};
    epilogue<NQ>(acc, base, so, nx, L, own, std::make_integer_sequence<int, 8 * NQ>{});
    if (!nt.nq && rfl(__hip_atomic_load((GAS unsigned*)(S.ws + OFF_TMO), __ATOMIC_RELAXED, __HIP_MEMORY_SCOPE_AGENT)) == call) S.bad = true;
    if (S.bad) {
      asm volatile("s_waitcnt vmcnt(0)" ::: "memory");
      *(volatile GAS float*)(base[0] + so) = __builtin_nanf("");
    }
    if (NQ == 3 && 2 * t.vp < OWNS) {
      if (own.raw) TELL("global_atomic_umax", L.line + OFF_PROG + (48 + p.rank) * 16, call);
      WAITV(0);
      TELL("global_atomic_add", L.line + OFF_RDY + 2 * 8192 + p.rank * 1024 + (t.pm & 7) * 128, 1u);
    }
  }
  if (ui < 2 && wid == 0)
    asm volatile("s_mov_b64 exec, 1\n\tglobal_atomic_add %0, %1, %2 sc1\n\ts_mov_b64 exec, -1" :: "v"(OFF_DONE + 8 + ui * 4), "v"(1u), "s"(S.ws) : "memory");
  if (nt.nq && !early) {
    if (!wait_progress(S.ws, call, need_batches(need_of(nt, S.level), min((nt.k0 + 3) >> 2, 15)))) { S.bad = true; S.all = true; }
    prologue(nt, XA, L);
  }
  return nt;
}
DI void gemm_role(const Params& p, LAS unsigned char* lds, unsigned x, unsigned r) {
  const int tid = threadIdx.x, wid = rfl(tid >> 6), lane = tid & 63, wr = wid >> 1, wc = wid & 1;
  asm volatile("" : "+s"(r));
  const Who w = {(x >> 1) & 1u, ((x >> 2) * 2 + (x & 1u)) * PER + r, r};
  const unsigned g = w.band * HB + w.s;
  const char* tell = pick(p.ws, lane & 7); asm volatile("" : "+v"(tell));
  const unsigned call = convert(p, upick(p.ws, p.rank), lds, tell, x, g * 4 + wid, wid, lane);
  Sync S; S.ws = upick(p.ws, p.rank); S.call = call; S.all = false; S.bad = false; S.tk = 0; S.junk = 0; S.level = 0; S.pw = 0;
  if (wid == 0) { queue_open(S.ws, OFF_TILEQ, call); queue_open(S.ws, OFF_PACE, call); }
  const char* XA = S.ws + CTRL;
  const unsigned lds0 = (unsigned)(size_t)lds;
  Lane L; L.voff = wid * 4096 + lane * 16; L.m0w = lds0 + wid * 4096; L.rA = lds0 + wr * 8192 + lane * 16; L.rB0 = lds0 + 65536 + wc * 8192 + lane * 16;
  L.line = tell;
  Tile t = tile_y(w.band, w.s);
  if (!wait_progress(S.ws, call, need_batches(need_of(t, 0), 0))) { S.bad = true; S.all = true; }
  prologue(t, XA, L);
  int ui = 0;
  do {
    while (t.nq == 3) t = run_tile<3>(p, lds, t, XA, L, S, ui++, wid, w);
    while (t.nq == 2) t = run_tile<2>(p, lds, t, XA, L, S, ui++, wid, w);
  } while (t.nq);
}

__global__ __launch_bounds__(256, 1) void fused(Params p) {
  extern __shared__ __attribute__((aligned(16))) unsigned char shm[];
  LAS unsigned char* lds = (LAS unsigned char*)shm;
  asm volatile(".Lgk_code_begin:");
  { unsigned junk = 0; code_prefetch(junk, (threadIdx.x >> 6) * 4096 + (threadIdx.x & 63) * 16); asm volatile("s_waitcnt vmcnt(0)" : "+v"(junk) :: "memory"); }
  unsigned bid = __builtin_amdgcn_workgroup_id_x();
  asm volatile("" : "+s"(bid));
  if (bid < NCOMM) comm_role(p, lds, bid, bid & 7u);
  else gemm_role(p, lds, bid & 7u, (bid >> 3) - NCOMM / 8);
}
}

extern "C" void kernel_launch(int rank, int nranks, ncclComm_t comm, hipStream_t stream,
                              void* const* d_in, const int* in_sizes, int n_in,
                              void* d_out, int out_size, const int* in_dtypes, int out_dtype,
                              void* const* d_out_all, void* const* d_ws_all, size_t ws_bytes) {
  using namespace gk;
  if (nranks != NR || (unsigned)rank >= (unsigned)NR || n_in != 2 || in_sizes[0] != M * KS || in_sizes[1] != KS * N || out_size != M * N ||
      ws_bytes < CTRL + BUF_BYTES + (size_t)M * N * 4) { fprintf(stderr, "kernel_launch: unexpected problem shape\n"); return; }
  static bool init = false;
  if (!init) { HIP_CHECK(hipFuncSetAttribute((const void*)fused, hipFuncAttributeMaxDynamicSharedMemorySize, LDS_BYTES)); init = true; }
  Params p; p.x = (const float*)d_in[0]; p.w = (const float*)d_in[1]; p.rank = rank; p.pad = 0;
  for (int i = 0; i < NR; ++i) { p.ws[i] = (char*)d_ws_all[i]; p.out[i] = (char*)d_out_all[i]; }
  fused<<<GRID, 256, LDS_BYTES, stream>>>(p);
  HIP_CHECK(hipGetLastError());
}
```
